```python
import numpy as np

kernel_name = "gpu_ag_gemm_m4096_k4096_n16384_e4m3e5m2_dqsilu"


ABSMAX_RTOL = 0.05

COMPUTE_DTYPE = "float32"
EPILOGUE = "dequant_silu"
IN_QUANT = None
OUT_QUANT = None
IN_FP8 = "e4m3e5m2"
IN_DTYPES = {"x": "e4m3", "w_mat": "e5m2",
             "scale_x": "float32", "scale_w": "float32"}
WORLD_SIZE = 8

def _snap_fp8(x, mant_bits, max_abs, min_normal):
    a = np.abs(x.astype(np.float64))
    _m, _e = np.frexp(a)
    q = np.where(a >= min_normal, np.ldexp(1.0, _e - 1 - mant_bits),
                 min_normal * 2.0 ** -mant_bits)
    snapped = np.minimum(np.round(a / q) * q, max_abs)
    return (np.sign(x) * snapped).astype(np.float32)

def setup_inputs():

    x = _snap_fp8(np.random.randn(4096, 4096).astype(np.float32), 3, 448.0, 2.0 ** -6)
    w_mat = _snap_fp8(np.random.randn(4096, 16384).astype(np.float32), 2, 57344.0, 2.0 ** -14)
    scale_x = (0.02 * (1.0 + np.random.rand(1))).astype(np.float32)
    scale_w = (0.02 * (1.0 + np.random.rand(1))).astype(np.float32)
    return {"x": x, "w_mat": w_mat, "scale_x": scale_x, "scale_w": scale_w}

def reference(x, w_mat, scale_x, scale_w):
    acc = (x @ w_mat).astype(np.float32)
    y = (acc * (scale_x * scale_w)).astype(np.float32)
    return (y / (1.0 + np.exp(-np.clip(y, -60.0, 60.0)))).astype(np.float32)
```

```cpp
#include <hip/hip_runtime.h>
#include <rccl/rccl.h>
#include <cstdio>

namespace {

typedef unsigned char u8;
typedef int v8i __attribute__((ext_vector_type(8)));
typedef int v4i __attribute__((ext_vector_type(4)));
typedef int v2i __attribute__((ext_vector_type(2)));
typedef unsigned u32x4 __attribute__((ext_vector_type(4)));
typedef float f32x4 __attribute__((ext_vector_type(4)));
#define LAS __attribute__((address_space(3)))
#define DI __device__ __forceinline__
#define GAP() __builtin_amdgcn_sched_barrier(0)

constexpr int NR = 8;
constexpr unsigned KD = 4096, NL = 2048, MSH = 512;
constexpr unsigned BLK = 65536;
constexpr int NRB = 32;
constexpr unsigned SHARD = NRB * BLK;
constexpr int GRID = 256, NCOMM = 32, NPUSH = 28;
constexpr unsigned OUT_BYTES = 4096u * NL * 4u;
constexpr size_t CNT_OFF = 0;
constexpr size_t TMO_OFF = 32768;
constexpr size_t FLAG_OFF = 65536;
constexpr unsigned FLAG_SRC = NRB * 128;
constexpr size_t DATA_OFF = 1u << 20;
constexpr size_t WS_NEED = DATA_OFF + 2 * NR * (size_t)SHARD;
constexpr unsigned LDS_BYTES = 131072;
constexpr unsigned long long DEADLINE = 60000000ull;
constexpr int AUX_SYS = 17;

struct Params {
  const u8* x; const u8* w; const float* sx; const float* sw; float* out;
  u8* myws; u8* ws[NR];
  int rank; int pad;
};

template <int N> DI void waitv() { asm volatile("s_waitcnt vmcnt(%0)" :: "i"(N) : "memory"); }
DI void ld16(u32x4& v, const void* p) { asm volatile("global_load_dwordx4 %0, %1, off" : "=v"(v) : "v"(p) : "memory"); }
DI void st16_sys(void* p, u32x4 v) { asm volatile("global_store_dwordx4 %0, %1, off sc0 sc1\n\ts_nop 1" :: "v"(p), "v"(v) : "memory"); }
DI void st4_sys(void* p, unsigned v) { asm volatile("global_store_dword %0, %1, off sc0 sc1" :: "v"(p), "v"(v) : "memory"); }
DI v4i ldx(__amdgpu_buffer_rsrc_t rs, unsigned voff, unsigned soff) {
  return __builtin_bit_cast(v4i, __builtin_amdgcn_raw_buffer_load_b128(rs, voff, soff, AUX_SYS));
}
DI bool flags_ready(u32x4 f, unsigned call) {
  unsigned m = min(min(f[0], f[1]), min(f[2], f[3]));
  return __builtin_amdgcn_readfirstlane(m) >= call;
}
DI float silu(float a, float scale) {
  float y = a * scale;
  float z = __builtin_amdgcn_fmed3f(y, -60.f, 60.f);
  float e = __builtin_amdgcn_exp2f(z * -1.44269504088896340736f);
  return y * __builtin_amdgcn_rcpf(1.f + e);
}

struct Gemm {
  v8i W[8][4];
  v4i A[2][16];
  f32x4 acc[2][4];
  u32x4 fl;
  const u8* x; const u8* slot; float* out; u8* tmo;
  __amdgpu_buffer_rsrc_t frs;
  unsigned vL, vR, vF, vO, wr[4], rd0, rd1;
  unsigned call, lstart, row_l, row_r, col4;
  int nl, nblk;
  float scale;
  bool bad;
};

DI void epi_write(Gemm& g, int pb) {
#pragma unroll
  for (int cb = 0; cb < 4; ++cb) *(LAS f32x4*)(size_t)(g.wr[cb] + pb * 4096) = g.acc[pb][cb];
}
DI void epi_barrier() { asm volatile("s_waitcnt lgkmcnt(0)\n\ts_barrier" ::: "memory"); }
DI void epi_read(Gemm& g, int pb, f32x4 (&p)[4]) {
  p[0] = *(LAS f32x4*)(size_t)(g.rd0 + pb * 4096);
  p[1] = *(LAS f32x4*)(size_t)(g.rd0 + pb * 4096 + 32768);
  p[2] = *(LAS f32x4*)(size_t)(g.rd1 + pb * 4096);
  p[3] = *(LAS f32x4*)(size_t)(g.rd1 + pb * 4096 + 32768);
}
DI void epi_store(Gemm& g, int j, f32x4 y) {
  const unsigned row0 = j < g.nl ? g.row_l + j * 16 : g.row_r + (j - g.nl) * 16;
  __amdgpu_buffer_rsrc_t ors = __builtin_amdgcn_make_buffer_rsrc((void*)g.out, 0, j >= 0 ? OUT_BYTES : 0u, 0x00020000);
  const unsigned soff = j >= 0 ? row0 * (NL * 4) + g.col4 : 0u;
  asm volatile("buffer_store_dwordx4 %0, %1, %2, %3 offen sc0 sc1\n\ts_nop 1" :: "v"(y), "v"(g.vO), "s"(ors), "s"(soff) : "memory");
}

template <int PB> DI void gemm_block(Gemm& g, const int j) {
  const int jl = j + 2, r = jl - g.nl;
  const bool rem = r >= 0, valid = jl < g.nblk;
  if (rem && valid && !g.bad && !flags_ready(g.fl, g.call)) {
    const unsigned long long t0 = __builtin_amdgcn_s_memrealtime();
    for (;;) {
      u32x4 f = __builtin_amdgcn_raw_buffer_load_b128(g.frs, g.vF, r * 128, AUX_SYS);
      if (flags_ready(f, g.call)) break;
      const unsigned t = __builtin_amdgcn_readfirstlane(__hip_atomic_load((unsigned*)g.tmo, __ATOMIC_RELAXED, __HIP_MEMORY_SCOPE_AGENT));
      if (t == g.call || __builtin_amdgcn_s_memrealtime() - t0 > DEADLINE) {
        asm volatile("global_store_dword %0, %1, off sc1\n\ts_waitcnt vmcnt(0)" :: "v"(g.tmo), "v"(g.call) : "memory");
        g.bad = true;
        g.scale = __builtin_nanf("");
        break;
      }
      __builtin_amdgcn_s_sleep(1);
    }
  }
  asm volatile("" ::: "memory");
  {
    const int rn = r + 1 < 0 ? 0 : (r + 1 > NRB - 1 ? NRB - 1 : r + 1);
    g.fl = __builtin_amdgcn_raw_buffer_load_b128(g.frs, g.vF, rn * 128, AUX_SYS);
  }
  __amdgpu_buffer_rsrc_t rs = __builtin_amdgcn_make_buffer_rsrc((void*)(rem ? g.slot : g.x), 0, (valid && !g.bad) ? SHARD : 0u, 0x00020000);
  const unsigned voff = rem ? g.vR : g.vL, step = rem ? 1024u : 64u;
  unsigned soff = valid ? (rem ? (unsigned)r : g.lstart + jl) * BLK : 0u;
  GAP();
  f32x4 p[4], y;
#pragma unroll
  for (int s = 0; s < 8; ++s) {
    const v8i a = __builtin_shufflevector(g.A[PB][2 * s], g.A[PB][2 * s + 1], 0, 1, 2, 3, 4, 5, 6, 7);
#pragma unroll
    for (int cb = 0; cb < 4; ++cb)
      g.acc[PB][cb] = __builtin_amdgcn_mfma_scale_f32_16x16x128_f8f6f4(g.W[s][cb], a, s == 0 ? f32x4{0.f, 0.f, 0.f, 0.f} : g.acc[PB][cb], 1, 0, 0, 0, 0, 0);
    g.A[PB][2 * s] = ldx(rs, voff, soff); soff += step;
    g.A[PB][2 * s + 1] = ldx(rs, voff, soff); soff += step;
    GAP();
    if (s == 0) { epi_write(g, PB ^ 1); GAP(); }
    if (s == 1) { epi_barrier(); epi_read(g, PB ^ 1, p); GAP(); }
    if (s == 3) { y = (p[0] + p[1]) + (p[2] + p[3]); y[0] = silu(y[0], g.scale); GAP(); }
    if (s == 4) { y[1] = silu(y[1], g.scale); y[2] = silu(y[2], g.scale); GAP(); }
    if (s == 5) { y[3] = silu(y[3], g.scale); epi_store(g, j - 1, y); GAP(); }
  }
}
template <int PB> DI void gemm_tail(Gemm& g) {
  f32x4 p[4], y;
  epi_write(g, PB); epi_barrier(); epi_read(g, PB, p);
  y = (p[0] + p[1]) + (p[2] + p[3]);
#pragma unroll
  for (int i = 0; i < 4; ++i) y[i] = silu(y[i], g.scale);
  epi_store(g, g.nblk - 1, y);
}

__global__ void __launch_bounds__(256, 1) ag_gemm(Params p) {
  extern __shared__ __attribute__((aligned(16))) u8 smem[];
  const unsigned lane = threadIdx.x & 63;
  const unsigned wave = __builtin_amdgcn_readfirstlane(threadIdx.x >> 6);
  const unsigned b = blockIdx.x;
  unsigned* cnt = (unsigned*)(p.myws + CNT_OFF + b * 128 + wave * 4);

  if (b < NCOMM) {
    if (b >= NPUSH) return;
    const unsigned link = b % 7, sub = (b / 7) * 4 + wave;
    const unsigned dstp = ((unsigned)p.rank + 1 + link) % NR;
    const u8* src = p.x + (lane & 15) * KD + sub * 256 + (lane >> 4) * 16;
    u32x4 d0[4], d1[4];
#pragma unroll
    for (int c = 0; c < 4; ++c) ld16(d0[c], src + c * 64);
#pragma unroll
    for (int c = 0; c < 4; ++c) ld16(d1[c], src + BLK + c * 64);
    unsigned call;
    asm volatile("global_load_dword %0, %1, off sc1\n\ts_waitcnt vmcnt(0)" : "=v"(call) : "v"(cnt) : "memory");
    call = __builtin_amdgcn_readfirstlane(call) + 1;
    u8* dws = p.ws[dstp];
    u8* dst = dws + DATA_OFF + (size_t)((call & 1) * NR + p.rank) * SHARD + sub * 4096 + lane * 16;
    u8* flg = dws + FLAG_OFF + p.rank * FLAG_SRC + sub * 4;
#define AG_STORE(D, S) _Pragma("unroll") for (int c = 0; c < 4; ++c) { asm volatile("" : "+v"(D[c])); st16_sys(dst + (size_t)(S) * BLK + c * 1024, D[c]); }
#define AG_LOAD(D, S) _Pragma("unroll") for (int c = 0; c < 4; ++c) ld16(D[c], src + (size_t)(S) * BLK + c * 64);
#define AG_FLAG(S) if (lane == 0) st4_sys(flg + (S) * 128, call);
    AG_STORE(d0, 0)
    AG_LOAD(d0, 2)
    AG_STORE(d1, 1)
    waitv<8>();
    AG_FLAG(0)
    for (int s = 2; s < NRB - 2; s += 2) {
      AG_LOAD(d1, s + 1)
      waitv<9>();
      AG_STORE(d0, s)
      waitv<9>();
      AG_FLAG(s - 1)
      AG_LOAD(d0, s + 2)
      waitv<9>();
      AG_STORE(d1, s + 1)
      waitv<9>();
      AG_FLAG(s)
    }
    AG_LOAD(d1, NRB - 1)
    waitv<9>();
    AG_STORE(d0, NRB - 2)
    waitv<9>();
    AG_FLAG(NRB - 3)
    waitv<5>();
    AG_STORE(d1, NRB - 1)
    waitv<5>();
    AG_FLAG(NRB - 2)
    waitv<0>();
    AG_FLAG(NRB - 1)
    if (lane == 0) asm volatile("global_store_dword %0, %1, off sc1" :: "v"(cnt), "v"(call) : "memory");
    return;
  }

  const unsigned gi = b - NCOMM, xcd = gi & 7, idx = gi >> 3, cg = idx & 3, rs = idx >> 2;
  const unsigned col0 = xcd * 256 + cg * 64;
  const unsigned srcp = ((unsigned)p.rank + 1 + rs) % NR;
  const unsigned lds = (unsigned)(size_t)smem + wave * 32768;
  Gemm g;
  unsigned callv;
  asm volatile("global_load_dword %0, %1, off sc1" : "=v"(callv) : "v"(cnt) : "memory");
  __amdgpu_buffer_rsrc_t wrs = __builtin_amdgcn_make_buffer_rsrc((void*)p.w, 0, KD * NL, 0x00020000);
  const unsigned wv0 = (lane >> 2) * NL + (((lane & 3) ^ ((lane >> 4) & 1)) * 16), wv1 = wv0 ^ 32;
  const unsigned ws0 = wave * 1024 * NL + col0;
  auto w_round = [&](int r) {
#pragma unroll
    for (int j = 0; j < 16; ++j)
      asm volatile("s_mov_b32 m0, %0\n\ts_nop 0\n\tbuffer_load_dwordx4 %1, %2, %3 offen lds"
                   :: "s"(lds + (r & 1) * 16384 + j * 1024), "v"((j & 1) ? wv1 : wv0), "s"(wrs), "s"(ws0 + (r * 256 + j * 16) * NL) : "memory");
  };
  const unsigned tg = lane >> 4, tr = (lane & 15) >> 1, tf = ((tr >> 2) & 1) | ((tg & 1) << 1);
  const unsigned tb = lds + (tg * 16 + tr) * 64 + tf * 16 + (lane & 1) * 8;
  auto w_read = [&](int r) {
#pragma unroll
    for (int h = 0; h < 2; ++h)
#pragma unroll
      for (int cb = 0; cb < 4; ++cb) {
        v2i t[4];
#pragma unroll
        for (int k = 0; k < 4; ++k)
          t[k] = __builtin_amdgcn_ds_read_tr8_b64_v2i32((LAS v2i*)(size_t)((tb ^ (cb * 16)) + (r & 1) * 16384 + (h * 128 + (k >> 1) * 64 + (k & 1) * 8) * 64));
        v8i& f = g.W[2 * r + h][cb];
        f = v8i{t[0][0], t[0][1], t[1][0], t[1][1], t[2][0], t[2][1], t[3][0], t[3][1]};
        if (2 * r + h < 7) asm volatile("" : "+a"(f)); else asm volatile("" : "+v"(f));
      }
  };
  w_round(0); w_round(1);
  g.x = p.x; g.out = p.out; g.tmo = p.myws + TMO_OFF;
  g.frs = __builtin_amdgcn_make_buffer_rsrc((void*)(p.myws + FLAG_OFF + srcp * FLAG_SRC), 0, FLAG_SRC, 0x00020000);
  g.vL = (lane & 15) * KD + (lane >> 4) * 16 + wave * 1024;
  g.vR = lane * 16 + wave * 16384;
  g.vF = wave * 16;
  g.vO = (wave * 4 + (lane >> 4)) * (NL * 4) + (lane & 15) * 16;
#pragma unroll
  for (int cb = 0; cb < 4; ++cb) g.wr[cb] = lds + (lane & 15) * 256 + (((cb * 4 + (lane >> 4)) ^ (lane & 15)) * 16);
  {
    const unsigned mr = wave * 4 + (lane >> 4);
    g.rd0 = (unsigned)(size_t)smem + mr * 256 + (((lane & 15) ^ mr) * 16);
    g.rd1 = g.rd0 + 65536;
  }
  g.nl = rs < 5 ? 4 : 6;
  g.lstart = rs < 5 ? 4 * rs : 6 * rs - 10;
  g.nblk = g.nl + NRB;
  g.row_l = p.rank * MSH + g.lstart * 16;
  g.row_r = srcp * MSH;
  g.col4 = col0 * 4;
  g.scale = *(const __attribute__((address_space(4))) float*)p.sx * *(const __attribute__((address_space(4))) float*)p.sw;
  g.bad = false;
  g.fl = u32x4{0u, 0u, 0u, 0u};
#pragma unroll
  for (int cb = 0; cb < 4; ++cb) g.acc[1][cb] = f32x4{0.f, 0.f, 0.f, 0.f};
  waitv<16>();
  asm volatile("" : "+v"(callv));
  g.call = __builtin_amdgcn_readfirstlane(callv) + 1;
  g.slot = p.myws + DATA_OFF + (size_t)((g.call & 1) * NR + srcp) * SHARD;
  w_read(0); w_round(2);
  waitv<16>();
  w_read(1); w_round(3);
  {
    __amdgpu_buffer_rsrc_t xr = __builtin_amdgcn_make_buffer_rsrc((void*)p.x, 0, SHARD, 0x00020000);
#pragma unroll
    for (int i = 0; i < 32; ++i) { GAP(); g.A[i >> 4][i & 15] = ldx(xr, g.vL, (g.lstart + (i >> 4)) * BLK + (i & 15) * 64); }
    GAP();
  }
  waitv<48>();
  w_read(2);
  waitv<32>();
  w_read(3);
  GAP();
  for (int j = 0; j < g.nblk; j += 2) {
    gemm_block<0>(g, j);
    gemm_block<1>(g, j + 1);
  }
  gemm_tail<1>(g);
  if (lane == 0) asm volatile("global_store_dword %0, %1, off sc1" :: "v"(cnt), "v"(g.call) : "memory");
}

}

#define HIP_CHECK(expr) do { hipError_t _e = (expr); if (_e != hipSuccess) { \
    fprintf(stderr, "HIP error %s at %s:%d\n", hipGetErrorString(_e), __FILE__, __LINE__); \
    return; } } while (0)

extern "C" void kernel_launch(int rank, int nranks, ncclComm_t comm, hipStream_t stream,
                              void* const* d_in, const int* in_sizes, int n_in,
                              void* d_out, int out_size, const int* in_dtypes, int out_dtype,
                              void* const* d_out_all, void* const* d_ws_all, size_t ws_bytes) {
  (void)comm; (void)in_dtypes; (void)out_dtype; (void)d_out_all;
  if (nranks != NR || (unsigned)rank >= (unsigned)NR || n_in < 4 || ws_bytes < WS_NEED ||
      in_sizes[0] != (int)(MSH * KD) || in_sizes[1] != (int)(KD * NL) || out_size != (int)(4096u * NL)) {
    fprintf(stderr, "kernel_launch: unexpected problem shape\n");
    return;
  }
  HIP_CHECK(hipFuncSetAttribute((const void*)ag_gemm, hipFuncAttributeMaxDynamicSharedMemorySize, LDS_BYTES));
  Params p;
  p.x = (const u8*)d_in[0]; p.w = (const u8*)d_in[1];
  p.sx = (const float*)d_in[2]; p.sw = (const float*)d_in[3];
  p.out = (float*)d_out;
  for (int i = 0; i < NR; ++i) p.ws[i] = (u8*)d_ws_all[i];
  p.myws = p.ws[rank];
  p.rank = rank; p.pad = 0;
  hipLaunchKernelGGL(ag_gemm, dim3(GRID), dim3(256), LDS_BYTES, stream, p);
  HIP_CHECK(hipGetLastError());
}
```

```cpp
#include <hip/hip_runtime.h>
#include <rccl/rccl.h>
#include <cstdio>

namespace {

typedef unsigned char u8;
typedef int v8i __attribute__((ext_vector_type(8)));
typedef int v4i __attribute__((ext_vector_type(4)));
typedef int v2i __attribute__((ext_vector_type(2)));
typedef unsigned u32x4 __attribute__((ext_vector_type(4)));
typedef float f32x4 __attribute__((ext_vector_type(4)));
#define LAS __attribute__((address_space(3)))
#define DI __device__ __forceinline__
#define GAP() __builtin_amdgcn_sched_barrier(0)

constexpr int NR = 8;
constexpr unsigned KD = 4096, NL = 2048, MSH = 512;
constexpr unsigned BLK = 65536;
constexpr int NRB = 32;
constexpr unsigned SHARD = NRB * BLK;
constexpr int GRID = 256, NCOMM = 32;
#define AG_G 2
#define AG_S2 24
#define AG_STB "sc0 sc1"
#define AG_X 0
#define AG_PN 4
#define AG_ROT 1
#define AG_SET 0x4d
#define AG_ST 26
constexpr int PN = AG_PN, PR = (PN + 3) / 4 + 1;
constexpr unsigned XG = AG_G;
constexpr unsigned OUT_BYTES = 4096u * NL * 4u;
constexpr size_t CNT_OFF = 0;
constexpr size_t TMO_OFF = 32768;
constexpr size_t FLAG_OFF = 65536;
constexpr unsigned FLAG_SRC = NRB * 128;
constexpr size_t DATA_OFF = 1u << 20;
constexpr size_t WS_NEED = DATA_OFF + 2 * NR * (size_t)SHARD;
constexpr unsigned LDS_BYTES = 131072;
constexpr unsigned DEADLINE = 60000000u;
constexpr int AUX_SYS = 17;
constexpr unsigned VOID_OFF = 0x20000000u;

struct Params {
  const u8* x; const u8* w; const float* sx; const float* sw; float* out;
  u8* myws; u8* peer[NR];
  int rank; int pad;
};

template <int N> DI void waitv() { asm volatile("s_waitcnt vmcnt(%0)" :: "i"(N) : "memory"); }
DI void ld16(u32x4& v, const void* p) { asm volatile("global_load_dwordx4 %0, %1, off" : "=v"(v) : "v"(p) : "memory"); }
DI void st16_sys(void* p, u32x4 v) { asm volatile("global_store_dwordx4 %0, %1, off sc0 sc1\n\ts_nop 1" :: "v"(p), "v"(v) : "memory"); }
DI void st4_sys(void* p, unsigned v) { asm volatile("global_store_dword %0, %1, off sc0 sc1" :: "v"(p), "v"(v) : "memory"); }
DI v4i ldx(__amdgpu_buffer_rsrc_t rs, unsigned voff, unsigned soff) {
  return __builtin_bit_cast(v4i, __builtin_amdgcn_raw_buffer_load_b128(rs, voff, soff, AUX_SYS));
}
DI bool flags_ready(u32x4 f, unsigned call) {
  unsigned m = min(min(f[0], f[1]), min(f[2], f[3]));
  return __builtin_amdgcn_readfirstlane(m) >= call;
}
DI float fadd(float a, float b) { float r; asm("v_add_f32 %0, %1, %2" : "=v"(r) : "v"(a), "v"(b)); return r; }
DI float fmul(float a, float b) { float r; asm("v_mul_f32 %0, %1, %2" : "=v"(r) : "v"(a), "v"(b)); return r; }
DI float fmuls(float a, float b) { float r; asm("v_mul_f32 %0, %2, %1" : "=v"(r) : "v"(a), "s"(b)); return r; }
DI float silu_a(float y) { return __builtin_amdgcn_exp2f(__builtin_amdgcn_fmed3f(y, -60.f, 60.f) * -1.44269504088896340736f); }
DI float silu_b(float e) { return __builtin_amdgcn_rcpf(1.f + e); }
template <int V> struct IC { static constexpr int value = V; };
template <int I, int N, class F> DI void static_for(F&& f) { if constexpr (I < N) { f(IC<I>{}); static_for<I + 1, N>(f); } }
DI int pin_s(int v) { asm volatile("" : "+s"(v)); return v; }

struct Gemm {
  v8i W[8][4];
  v4i A[2][16];
  f32x4 acc[2][4];
  u32x4 f;
  const u8* slots;
  __amdgpu_buffer_rsrc_t frs, ars, ors;
  unsigned long long lanes;
  unsigned vR, vO, vF, tab, so1, so2, wr[4], rd0, rd1;
  unsigned call, t0, blk_l, blk_r, col4;
  int nl, nblk, j2;
  float scale;
  unsigned live, need;
};

DI unsigned gblk(const Gemm& g, int j) { return j + (j < g.nl ? g.blk_l : g.blk_r); }
DI void epi_write(Gemm& g, int pb) {
#pragma unroll
  for (int cb = 0; cb < 4; ++cb) *(LAS f32x4*)(size_t)(g.wr[cb] + pb * 4096) = g.acc[pb][cb];
}
DI void epi_barrier() { asm volatile("s_waitcnt lgkmcnt(0)\n\ts_barrier" ::: "memory"); }
DI void epi_read(Gemm& g, int pb, f32x4 (&p)[4]) {
  p[0] = *(LAS f32x4*)(size_t)(g.rd0 + pb * 4096);
  p[1] = *(LAS f32x4*)(size_t)(g.rd0 + pb * 4096 + 32768);
  p[2] = *(LAS f32x4*)(size_t)(g.rd1 + pb * 4096);
  p[3] = *(LAS f32x4*)(size_t)(g.rd1 + pb * 4096 + 32768);
}
DI void epi_store(Gemm& g, unsigned soff, f32x4 y) {
  asm volatile("buffer_store_dwordx4 %0, %1, %2, %3 offen " AG_STB "\n\ts_nop 1" :: "v"(y), "v"(g.vO), "s"(g.ors), "s"(soff) : "memory");
}

DI void sample(Gemm& g, unsigned long long lanes) {
  asm volatile("s_mov_b64 exec, %2\n\tbuffer_load_dwordx4 %0, %1, %3, 0 offen sc0 sc1\n\ts_mov_b64 exec, -1"
               :: "v"(g.f), "v"(g.vF), "s"(lanes), "s"(g.frs) : "memory");
}
DI unsigned long long ready(Gemm& g) {
  unsigned long long m; unsigned t;
  asm volatile("v_min_u32 %1, %2, %3\n\tv_min3_u32 %1, %1, %4, %5\n\tv_cmp_le_u32 %0, %6, %1"
               : "=s"(m), "=&v"(t) : "v"(g.f[0]), "v"(g.f[1]), "v"(g.f[2]), "v"(g.f[3]), "s"(g.call) : "memory");
  return m;
}
DI void wait_step(Gemm& g, int js) {
  unsigned long long lanes; unsigned t;
  asm volatile("s_cmp_eq_u32 %2, 0\n\ts_cbranch_scc1 .Ldone%=\n\t"
               "s_lshl_b64 %0, 3, %13\n"
               ".Lpoll%=:\n\t"
               "v_min_u32 %1, %6, %7\n\tv_min3_u32 %1, %1, %8, %9\n\tv_cmp_le_u32 vcc, %12, %1\n\t"
               "s_bitcmp1_b64 vcc, %13\n\ts_cbranch_scc1 .Ldone%=\n\t"
               "s_mov_b64 exec, %0\n\tbuffer_load_dwordx4 %5, %10, %11, 0 offen sc0 sc1\n\ts_mov_b64 exec, -1\n\t"
               "s_memrealtime vcc\n\ts_waitcnt lgkmcnt(0)\n\t"
               "s_sub_u32 vcc_lo, vcc_lo, %14\n\ts_cmp_lt_u32 vcc_lo, %15\n\ts_cbranch_scc1 .Lpoll%=\n\t"
               "s_mov_b32 %2, 0\n\ts_mov_b32 %3, 0\n\ts_mov_b32 %4, 0x7fc00000\n"
               ".Ldone%=:"
               : "=&s"(lanes), "=&v"(t), "+s"(g.need), "+s"(g.live), "+s"(g.scale)
               : "v"(g.f), "v"(g.f[0]), "v"(g.f[1]), "v"(g.f[2]), "v"(g.f[3]), "v"(g.vF), "s"(g.frs), "s"(g.call), "s"(js),
                 "s"(g.t0), "i"(DEADLINE) : "memory", "vcc", "scc");
}
#define AG_LD(k, o) "buffer_load_dwordx4 %" #k ", %17, %18, %16 offen offset:" #o " sc0 sc1\n\t"
#define AG_LD4(a, b, c, d) AG_LD(a, 0) AG_LD(b, 1024) AG_LD(c, 2048) AG_LD(d, 3072)
#define AG_NEXT "s_add_u32 %16, %16, 0x1000\n\t"
DI void load_block(Gemm& g, v4i (&A)[16], unsigned so, unsigned need) {
  asm volatile("s_cmp_eq_u32 %19, 0\n\ts_cbranch_scc1 .Lskip%=\n\t"
               AG_LD4(0, 1, 2, 3) AG_NEXT AG_LD4(4, 5, 6, 7) AG_NEXT AG_LD4(8, 9, 10, 11) AG_NEXT AG_LD4(12, 13, 14, 15) "\n.Lskip%=:"
               : "+v"(A[0]), "+v"(A[1]), "+v"(A[2]), "+v"(A[3]), "+v"(A[4]), "+v"(A[5]), "+v"(A[6]), "+v"(A[7]),
                 "+v"(A[8]), "+v"(A[9]), "+v"(A[10]), "+v"(A[11]), "+v"(A[12]), "+v"(A[13]), "+v"(A[14]), "+v"(A[15]), "+s"(so)
               : "v"(g.vR), "s"(g.ars), "s"(need) : "memory", "scc");
}
template <int O> DI void load_one(Gemm& g, v4i& a, unsigned so) {
  asm volatile("buffer_load_dwordx4 %0, %1, %2, %3 offen offset:%4 sc0 sc1" : "+v"(a) : "v"(g.vR), "s"(g.ars), "s"(so), "i"(O) : "memory");
}

template <int PB> DI void gemm_block(Gemm& g) {
  unsigned so, sst;
  unsigned long long m;
  f32x4 p[4], y, e;
  wait_step(g, g.j2 - 1);
  GAP();
  load_block(g, g.A[PB ^ 1], g.so1, g.need);
  GAP();
  static_for<0, 32>([&](auto tc) __attribute__((always_inline)) {
    constexpr int t = decltype(tc)::value, s = t >> 2, cb = t & 3;
    if constexpr (cb == 0) asm volatile("s_waitcnt vmcnt(%2)" : "+v"(g.A[PB][2 * s]), "+v"(g.A[PB][2 * s + 1]) : "i"(AG_S2 < 32 ? 33 : 32) : "memory");
    const v8i a = __builtin_shufflevector(g.A[PB][2 * s], g.A[PB][2 * s + 1], 0, 1, 2, 3, 4, 5, 6, 7);
    g.acc[PB][cb] = __builtin_amdgcn_mfma_scale_f32_16x16x128_f8f6f4(g.W[s][cb], a, s == 0 ? f32x4{0.f, 0.f, 0.f, 0.f} : g.acc[PB][cb], 1, 0, 0, 0, 0, 0);
    GAP();
    if constexpr (t == 0) m = ready(g);
    if constexpr (t == 1) { g.lanes <<= 1; sample(g, g.lanes); }
    if constexpr (t == AG_S2) sample(g, g.lanes >> 1);
    if constexpr (t == 2) {
      g.so1 = g.so2;
      asm volatile("s_bitcmp1_b64 %2, %3\n\ts_cselect_b32 %0, %4, %5\n\ts_cselect_b32 %1, 0, %6"
                   : "=s"(so), "=s"(g.need) : "s"(m), "s"(g.j2), "s"(g.so1), "i"(VOID_OFF), "s"(g.live) : "scc");
    }
    if constexpr (t == 28) { g.j2 = pin_s(g.j2) + 1; g.so2 = pin_s(__builtin_amdgcn_readlane(g.tab, g.j2)); }
    if constexpr (t == 2 || t == 4) {
      *(LAS f32x4*)(size_t)(g.wr[t - 2] + (PB ^ 1) * 4096) = g.acc[PB ^ 1][t - 2];
      *(LAS f32x4*)(size_t)(g.wr[t - 1] + (PB ^ 1) * 4096) = g.acc[PB ^ 1][t - 1];
    }
    if constexpr (t >= 3 && (t & 1)) {
      constexpr int i = (t - 3) >> 1;
      if constexpr (i && !(i & 3)) so = pin_s(so) + 4096;
      load_one<(i & 3) * 1024>(g, g.A[PB][i], so);
      if constexpr (t == 31) load_one<3072>(g, g.A[PB][15], so);
    }
    if constexpr (t == 8) { epi_barrier(); p[0] = *(LAS f32x4*)(size_t)(g.rd0 + (PB ^ 1) * 4096); p[1] = *(LAS f32x4*)(size_t)(g.rd0 + (PB ^ 1) * 4096 + 32768); }
    if constexpr (t == 10) { p[2] = *(LAS f32x4*)(size_t)(g.rd1 + (PB ^ 1) * 4096); p[3] = *(LAS f32x4*)(size_t)(g.rd1 + (PB ^ 1) * 4096 + 32768); }
    if constexpr (t == 12) { _Pragma("unroll") for (int i = 0; i < 4; ++i) p[0][i] = fadd(p[0][i], p[1][i]); }
    if constexpr (t == 14) { _Pragma("unroll") for (int i = 0; i < 4; ++i) p[2][i] = fadd(p[2][i], p[3][i]); }
    if constexpr (t == 15) { _Pragma("unroll") for (int i = 0; i < 4; ++i) p[0][i] = fadd(p[0][i], p[2][i]); }
    if constexpr (t == 16) { _Pragma("unroll") for (int i = 0; i < 4; ++i) y[i] = fmuls(p[0][i], g.scale); }
    if constexpr (t >= 17 && t <= 20) e[t - 17] = silu_a(y[t - 17]);
    if constexpr (t >= 22 && t <= 25) y[t - 22] = fmul(y[t - 22], e[t - 22]);
    if constexpr (t >= 21 && t <= 24) e[t - 21] = silu_b(e[t - 21]);
    if constexpr (t == 6) sst = pin_s(__builtin_amdgcn_readlane(g.tab, pin_s(g.j2) - 3) * 2 + g.col4);
    if constexpr (t == AG_ST) epi_store(g, sst, y);
    GAP();
  });
}
template <int PB> DI void gemm_tail(Gemm& g) {
  f32x4 p[4], y;
  epi_write(g, PB); epi_barrier(); epi_read(g, PB, p);
#pragma unroll
  for (int i = 0; i < 4; ++i) {
    y[i] = ((p[0][i] + p[1][i]) + (p[2][i] + p[3][i])) * g.scale;
    y[i] *= silu_b(silu_a(y[i]));
  }
  epi_store(g, gblk(g, g.nblk - 1) * (2 * BLK) + g.col4, y);
}

__global__ void __launch_bounds__(256, 1) ag_gemm(Params p) {
  extern __shared__ __attribute__((aligned(16))) u8 smem[];
  const unsigned lane = threadIdx.x & 63;
  const unsigned wave = __builtin_amdgcn_readfirstlane(threadIdx.x >> 6);
  const unsigned b = blockIdx.x;
  u8* const dws = p.peer[b & 7];
  asm volatile("" :: "s"(p.x), "s"(p.w), "s"(p.sx), "s"(p.sw), "s"(p.out), "s"(p.myws), "s"(p.rank), "s"(dws));
  unsigned* cnt = (unsigned*)(p.myws + CNT_OFF + b * 128 + wave * 4);
  const unsigned piece = b < NCOMM ? (b >> 3) * 4 + wave : ((b - NCOMM) & 127) * 4 + wave, prb = piece >> 4, psub = piece & 15;
  const u8* src = p.x + (prb * BLK + psub * 256) + ((lane & 15) * KD + (lane >> 4) * 16);
  u32x4 xs[4];
#pragma unroll
  for (int c = 0; c < 4; ++c) ld16(xs[c], src + c * 64);
  unsigned callv;
  asm volatile("global_load_dword %0, %1, off sc1" : "=v"(callv) : "v"(cnt) : "memory");

  if (b < NCOMM) {
    if (dws == p.myws) {
      unsigned call;
      asm volatile("s_waitcnt vmcnt(0)\n\tv_add_u32 %0, 1, %1" : "=v"(call) : "v"(callv), "v"(xs[0]), "v"(xs[1]), "v"(xs[2]), "v"(xs[3]) : "memory");
      if (lane == 0) asm volatile("global_store_dword %0, %1, off sc1" :: "v"(cnt), "v"(call) : "memory");
      return;
    }
    const __amdgpu_buffer_rsrc_t xrs = __builtin_amdgcn_make_buffer_rsrc((void*)p.x, 0, SHARD, 0x00020000);
    const __amdgpu_buffer_rsrc_t frs = __builtin_amdgcn_make_buffer_rsrc((void*)(dws + FLAG_OFF + p.rank * FLAG_SRC), 0, FLAG_SRC, 0x00020000);
    const unsigned vx = (lane & 15) * KD + (lane >> 4) * 16, vd = lane * 16;
    u32x4 d[PR][4];
    auto sub = [&](int i) { return (psub + AG_ROT * i) & 15; };
    auto load = [&](u32x4 (&v)[4], int i) {
      asm volatile("buffer_load_dwordx4 %0, %4, %5, %6 offen\n\tbuffer_load_dwordx4 %1, %4, %5, %6 offen offset:64\n\t"
                   "buffer_load_dwordx4 %2, %4, %5, %6 offen offset:128\n\tbuffer_load_dwordx4 %3, %4, %5, %6 offen offset:192"
                   : "=&v"(v[0]), "=&v"(v[1]), "=&v"(v[2]), "=&v"(v[3]) : "v"(vx), "s"(xrs), "s"(i * BLK + sub(i) * 256) : "memory");
    };
#pragma unroll
    for (int i = 1; i < PR; ++i) load(d[i], i);
    unsigned call;
    asm volatile("s_waitcnt vmcnt(4)\n\tv_readfirstlane_b32 %0, %1" : "=s"(call) : "v"(callv) : "memory");
#pragma unroll
    for (int c = 0; c < 4; ++c)
      asm volatile("v_mov_b64 %0, %2\n\tv_mov_b64 %1, %3" : "=&v"(*(v2i*)&d[0][c]), "=&v"(*((v2i*)&d[0][c] + 1)) : "v"(*(v2i*)&xs[c]), "v"(*((v2i*)&xs[c] + 1)));
    call += 1;
    const __amdgpu_buffer_rsrc_t drs = __builtin_amdgcn_make_buffer_rsrc((void*)(dws + DATA_OFF + (size_t)((call & 1) * NR + p.rank) * SHARD), 0, SHARD, 0x00020000);
    unsigned callx = call;
    asm volatile("" : "+v"(callx));
#pragma unroll 1
    for (int i0 = 0; i0 < NRB + PR - 1; i0 += PR) {
      static_for<0, 4 * PR>([&](auto kc) __attribute__((always_inline)) {
        constexpr int r = decltype(kc)::value >> 2, k = decltype(kc)::value & 3;
        constexpr int kf = (PN + 3) & 3, back = (PN - k + 3) >> 2;
        constexpr int flags = (PN - 1 - ((k - kf + 3) & 3) + 3) >> 2;
        const int i = i0 + r;
        waitv<PN - 1 + 4 * back + flags>();
        if constexpr (k == kf) {
          asm volatile("s_mov_b64 exec, 1\n\tbuffer_store_dword %0, off, %1, %2 sc0 sc1\n\ts_mov_b64 exec, -1"
                       :: "v"(callx), "s"(frs), "s"((i - back) * 128 + sub(i - back) * 4) : "memory");
        }
        asm volatile("buffer_store_dwordx4 %0, %1, %2, %3 offen offset:%4 sc0 sc1\n\ts_nop 1"
                     :: "v"(d[r][k]), "v"(vd), "s"(drs), "s"(i * BLK + sub(i) * 4096), "i"(k * 1024) : "memory");
        if constexpr (k == 3) load(d[r], i + PR);
      });
    }
    waitv<0>();
#pragma unroll
    for (int i = 0; i < PR; ++i) asm volatile("" :: "v"(d[i][0]), "v"(d[i][1]), "v"(d[i][2]), "v"(d[i][3]));
    if (lane == 0) asm volatile("global_store_dword %0, %1, off sc1" :: "v"(cnt), "v"(call) : "memory");
    return;
  }

  const unsigned gi = b - NCOMM, xcd = gi & 7, pos = (xcd % XG) * 28 + (gi >> 3), rs = pos / (4 * XG);
  const unsigned col0 = ((xcd / XG) * 4 * XG + pos % (4 * XG)) * 64;
  const unsigned srcp = ((unsigned)p.rank + 1 + rs) % NR;
  const unsigned lds = (unsigned)(size_t)smem + wave * 32768;
  Gemm g;
  g.t0 = (unsigned)__builtin_amdgcn_s_memrealtime();
  __amdgpu_buffer_rsrc_t wrs = __builtin_amdgcn_make_buffer_rsrc((void*)p.w, 0, KD * NL, 0x00020000);
  const unsigned wv0 = (lane >> 2) * NL + (((lane & 3) ^ ((lane >> 4) & 1)) * 16), wv1 = wv0 ^ 32;
  const unsigned ws0 = wave * 1024 * NL + col0;
  const unsigned rot = rs * 4;
  auto w_dma = [&](int j0) {
#pragma unroll
    for (int j = j0; j < j0 + 16; ++j) {
      const unsigned q = (j + rot) & 31;
      asm volatile("s_mov_b32 m0, %0\n\ts_nop 0\n\tbuffer_load_dwordx4 %1, %2, %3 offen lds"
                   :: "s"(lds + q * 1024), "v"((j & 1) ? wv1 : wv0), "s"(wrs), "s"(ws0 + q * 16 * NL) : "memory");
    }
  };
  const unsigned tg = lane >> 4, tr = (lane & 15) >> 1, tf = ((tr >> 2) & 1) | ((tg & 1) << 1);
  const unsigned tb = lds + (tg * 16 + tr) * 64 + tf * 16 + (lane & 1) * 8;
  auto w_read = [&](int r) {
#pragma unroll
    for (int h = 0; h < 2; ++h)
#pragma unroll
      for (int cb = 0; cb < 4; ++cb) {
        v2i t[4];
#pragma unroll
        for (int k = 0; k < 4; ++k)
          t[k] = __builtin_amdgcn_ds_read_tr8_b64_v2i32((LAS v2i*)(size_t)((tb ^ (cb * 16)) + (r & 1) * 16384 + (h * 128 + (k >> 1) * 64 + (k & 1) * 8) * 64));
        g.W[2 * r + h][cb] = v8i{t[0][0], t[0][1], t[1][0], t[1][1], t[2][0], t[2][1], t[3][0], t[3][1]};
      }
#pragma unroll
    for (int h = 0; h < 2; ++h)
#pragma unroll
      for (int cb = 0; cb < 4; ++cb) {
        v8i& f = g.W[2 * r + h][cb];
        if (2 * r + h < 7) asm volatile("" : "+a"(f)); else asm volatile("" : "+v"(f));
      }
  };
  w_dma(0);
  g.ors = __builtin_amdgcn_make_buffer_rsrc((void*)p.out, 0, (AG_X & 1) ? 0 : OUT_BYTES, 0x00020000);
  g.frs = __builtin_amdgcn_make_buffer_rsrc((void*)p.myws, 0, FLAG_OFF + NR * FLAG_SRC, 0x00020000);
  g.vR = lane * 16 + wave * 16384;
  g.vO = (wave * 4 + (lane >> 4)) * (NL * 4) + (lane & 15) * 16;
#pragma unroll
  for (int cb = 0; cb < 4; ++cb) g.wr[cb] = lds + (lane & 15) * 256 + (((cb * 4 + (lane >> 4)) ^ (lane & 15)) * 16);
  {
    const unsigned mr = wave * 4 + (lane >> 4);
    g.rd0 = (unsigned)(size_t)smem + mr * 256 + (((lane & 15) ^ mr) * 16);
    g.rd1 = g.rd0 + 65536;
    asm volatile("" : "+v"(g.rd1));
  }
  g.nl = rs < 5 ? 4 : 6;
  g.nblk = g.nl + NRB;
  g.blk_l = p.rank * NRB + (rs < 5 ? 4 * rs : 6 * rs - 10);
  g.blk_r = srcp * NRB - g.nl;
  g.col4 = col0 * 4;
  g.vF = FLAG_OFF + gblk(g, (int)lane < g.nblk ? lane : 0) * 128 + wave * 16;
  g.tab = (int)lane < g.nblk ? gblk(g, lane) * BLK : VOID_OFF;
  g.f = u32x4{0, 0, 0, 0};
  asm volatile("" : "+v"(g.f));
  g.scale = __builtin_bit_cast(float, __builtin_amdgcn_readfirstlane(__builtin_bit_cast(int,
      *(const __attribute__((address_space(4))) float*)p.sx * *(const __attribute__((address_space(4))) float*)p.sw)));
  g.live = 1; g.need = 1;
#pragma unroll
  for (int cb = 0; cb < 4; ++cb) g.acc[1][cb] = f32x4{0.f, 0.f, 0.f, 0.f};
#define WSTAMP(i)
  waitv<16>();
  WSTAMP(0)
  asm volatile("v_readfirstlane_b32 %0, %1" : "=s"(g.call) : "v"(callv) : "memory");
  g.call += 1;
  g.slots = p.myws + DATA_OFF + (size_t)((g.call & 1) * NR) * SHARD;
  g.ars = __builtin_amdgcn_make_buffer_rsrc((void*)g.slots, 0, NR * SHARD, 0x00020000);
  {
    __amdgpu_buffer_rsrc_t prs = __builtin_amdgcn_make_buffer_rsrc((void*)g.slots, 0, gi < 128 ? NR * SHARD : 0u, 0x00020000);
#pragma unroll
    for (int c = 0; c < 4; ++c)
      asm volatile("buffer_store_dwordx4 %0, %1, %2, %3 offen sc0 sc1\n\ts_nop 1"
                   :: "v"(xs[c]), "v"(lane * 16 + c * 1024), "s"(prs), "s"((p.rank * NRB + prb) * BLK + psub * 4096) : "memory");
  }
  w_dma(16);
  v4i wr[32];
#pragma unroll
  for (int j = 0; j < 32; ++j) {
    GAP();
    wr[j] = __builtin_bit_cast(v4i, __builtin_amdgcn_raw_buffer_load_b128(wrs, (j & 1) ? wv1 : wv0, ws0 + (32 + ((j + rot) & 31)) * 16 * NL, 0));
  }
  GAP();
  waitv<48>();
  WSTAMP(1)
  if (lane == 0) {
    __amdgpu_buffer_rsrc_t prs = __builtin_amdgcn_make_buffer_rsrc((void*)(p.myws + FLAG_OFF), 0, gi < 128 ? NR * FLAG_SRC : 0u, 0x00020000);
    asm volatile("buffer_store_dword %0, off, %1, %2 sc0 sc1" :: "v"(g.call), "s"(prs), "s"((p.rank * NRB + prb) * 128 + psub * 4) : "memory");
  }
  waitv<33>();
  WSTAMP(2)
  w_read(0); w_read(1);
  WSTAMP(3)
#pragma unroll
  for (int j = 0; j < 32; ++j) *(LAS v4i*)(size_t)(lds + ((j + rot) & 31) * 1024 + lane * 16) = wr[j];
  WSTAMP(4)
  wait_step(g, 0);
  wait_step(g, 1);
  WSTAMP(5)
  g.lanes = 3ull << 2;
  sample(g, g.lanes);
#pragma unroll
  for (int i = 0; i < 16; ++i) { asm volatile("" : "=v"(g.A[0][i])); asm volatile("" : "=v"(g.A[1][i])); }
  load_block(g, g.A[0], g.blk_l * BLK, g.live);
  load_block(g, g.A[1], (g.blk_l + 1) * BLK, g.live);
  g.need = 0; g.so1 = 0; g.so2 = (g.blk_l + 2) * BLK; g.j2 = 2;
  w_read(2); w_read(3);
  waitv<16>();
  GAP();
  const int jend = g.nblk + 2;
  do {
    gemm_block<0>(g);
    gemm_block<1>(g);
  } while (g.j2 < jend);
  gemm_tail<1>(g);
  if (lane == 0) {
    asm volatile("global_store_dword %0, %1, off sc1" :: "v"(cnt), "v"(g.call) : "memory");
    if (!g.live) asm volatile("global_store_dword %0, %1, off sc1" :: "v"(p.myws + TMO_OFF), "v"(g.call) : "memory");
  }
  asm volatile("" :: "v"(g.f));
#pragma unroll
  for (int i = 0; i < 16; ++i) asm volatile("" :: "v"(g.A[0][i]), "v"(g.A[1][i]));
}

}

#define HIP_CHECK(expr) do { hipError_t _e = (expr); if (_e != hipSuccess) { \
    fprintf(stderr, "HIP error %s at %s:%d\n", hipGetErrorString(_e), __FILE__, __LINE__); \
    return; } } while (0)

extern "C" void kernel_launch(int rank, int nranks, ncclComm_t comm, hipStream_t stream,
                              void* const* d_in, const int* in_sizes, int n_in,
                              void* d_out, int out_size, const int* in_dtypes, int out_dtype,
                              void* const* d_out_all, void* const* d_ws_all, size_t ws_bytes) {
  (void)comm; (void)in_dtypes; (void)out_dtype; (void)d_out_all;
  if (nranks != NR || (unsigned)rank >= (unsigned)NR || n_in < 4 || ws_bytes < WS_NEED ||
      in_sizes[0] != (int)(MSH * KD) || in_sizes[1] != (int)(KD * NL) || out_size != (int)(4096u * NL)) {
    fprintf(stderr, "kernel_launch: unexpected problem shape\n");
    return;
  }
  HIP_CHECK(hipFuncSetAttribute((const void*)ag_gemm, hipFuncAttributeMaxDynamicSharedMemorySize, LDS_BYTES));
  Params p;
  p.x = (const u8*)d_in[0]; p.w = (const u8*)d_in[1];
  p.sx = (const float*)d_in[2]; p.sw = (const float*)d_in[3];
  p.out = (float*)d_out;
  for (int i = 0, lo = 0, hi = 4; i < NR; ++i) {
    const int r = (rank + 1 + i) % NR;
    p.peer[(((AG_SET >> r) ^ (AG_SET >> rank)) & 1) ? hi++ : lo++] = (u8*)d_ws_all[r];
  }
  p.myws = (u8*)d_ws_all[rank];
  p.rank = rank; p.pad = 0;
  hipLaunchKernelGGL(ag_gemm, dim3(GRID), dim3(256), LDS_BYTES, stream, p);
  HIP_CHECK(hipGetLastError());
}
```

```cpp
#include <hip/hip_runtime.h>
#include <rccl/rccl.h>
#include <cstdio>

namespace {

typedef unsigned char u8;
typedef int v8i __attribute__((ext_vector_type(8)));
typedef int v4i __attribute__((ext_vector_type(4)));
typedef int v2i __attribute__((ext_vector_type(2)));
typedef unsigned u32x4 __attribute__((ext_vector_type(4)));
typedef float f32x4 __attribute__((ext_vector_type(4)));
#define LAS __attribute__((address_space(3)))
#define DI __device__ __forceinline__
#define GAP() __builtin_amdgcn_sched_barrier(0)

constexpr int NR = 8;
constexpr unsigned KD = 4096, NL = 2048, MSH = 512;
constexpr unsigned BLK = 65536;
constexpr int NRB = 32;
constexpr unsigned SHARD = NRB * BLK;
constexpr int GRID = 256, NCOMM = 32;
constexpr unsigned NPW = 3;
constexpr unsigned GPU_SET = 0x4d;
constexpr unsigned long long LOCAL0 = 0x201c18130f0a0500ull;
constexpr unsigned OUT_BYTES = 4096u * NL * 4u;
constexpr size_t CNT_OFF = 0;
constexpr size_t TMO_OFF = 32768;
constexpr size_t FLAG_OFF = 65536;
constexpr unsigned FQ = 128, FB = 4 * FQ, FLAG_SRC = NRB * FB;
constexpr size_t TICK_OFF = FLAG_OFF + NR * FLAG_SRC;
constexpr size_t DATA_OFF = 1u << 20;
constexpr size_t WS_NEED = DATA_OFF + 2 * NR * (size_t)SHARD;
constexpr unsigned LDS_BYTES = 131072;
constexpr unsigned DEADLINE = 60000000u;
constexpr unsigned VOID_OFF = 0x20000000u;

struct Params {
  const u8* x; const u8* w; const float* sx; const float* sw; float* out;
  u8* myws; u8* peer[NR];
  int rank; unsigned ranks;
  unsigned map[GRID / 4];
};

template <int N> DI void waitv() { asm volatile("s_waitcnt vmcnt(%0)" :: "i"(N) : "memory"); }
DI float fadd(float a, float b) { float r; asm("v_add_f32 %0, %1, %2" : "=v"(r) : "v"(a), "v"(b)); return r; }
DI float fmul(float a, float b) { float r; asm("v_mul_f32 %0, %1, %2" : "=v"(r) : "v"(a), "v"(b)); return r; }
DI float fmuls(float a, float b) { float r; asm("v_mul_f32 %0, %2, %1" : "=v"(r) : "v"(a), "s"(b)); return r; }
DI float silu_a(float y) { return __builtin_amdgcn_exp2f(__builtin_amdgcn_fmed3f(y, -60.f, 60.f) * -1.44269504088896340736f); }
DI float silu_b(float e) { return __builtin_amdgcn_rcpf(1.f + e); }
template <int V> struct IC { static constexpr int value = V; };
template <int I, int N, class F> DI void static_for(F&& f) { if constexpr (I < N) { f(IC<I>{}); static_for<I + 1, N>(f); } }
DI int pin_s(int v) { asm volatile("" : "+s"(v)); return v; }

struct Gemm {
  v8i W[8][4];
  v4i A[2][16];
  f32x4 acc[2][4];
  u32x4 f;
  const u8* slots;
  __amdgpu_buffer_rsrc_t frs, ars, ors;
  unsigned long long lanes;
  unsigned vR, vO, vF, tab, so1, so2, wr[4], rd0, rd1;
  unsigned call, t0, blk_l, blk_r, col4;
  int nl, nblk, j2;
  float scale;
  unsigned live, need;
};

DI unsigned gblk(const Gemm& g, int j) { return j + (j < g.nl ? g.blk_l : g.blk_r); }
DI void epi_write(Gemm& g, int pb) {
#pragma unroll
  for (int cb = 0; cb < 4; ++cb) *(LAS f32x4*)(size_t)(g.wr[cb] + pb * 4096) = g.acc[pb][cb];
}
DI void epi_barrier() { asm volatile("s_waitcnt lgkmcnt(0)\n\ts_barrier" ::: "memory"); }
DI void epi_read(Gemm& g, int pb, f32x4 (&p)[4]) {
  p[0] = *(LAS f32x4*)(size_t)(g.rd0 + pb * 4096);
  p[1] = *(LAS f32x4*)(size_t)(g.rd0 + pb * 4096 + 32768);
  p[2] = *(LAS f32x4*)(size_t)(g.rd1 + pb * 4096);
  p[3] = *(LAS f32x4*)(size_t)(g.rd1 + pb * 4096 + 32768);
}
DI void epi_store(Gemm& g, unsigned soff, f32x4 y) {
  asm volatile("buffer_store_dwordx4 %0, %1, %2, %3 offen sc0 sc1\n\ts_nop 1" :: "v"(y), "v"(g.vO), "s"(g.ors), "s"(soff) : "memory");
}

DI void sample(Gemm& g, unsigned long long lanes) {
  asm volatile("s_mov_b64 exec, %2\n\tbuffer_load_dwordx4 %0, %1, %3, 0 offen sc0 sc1\n\ts_mov_b64 exec, -1"
               :: "v"(g.f), "v"(g.vF), "s"(lanes), "s"(g.frs) : "memory");
}
DI unsigned long long ready(Gemm& g) {
  unsigned long long m; unsigned t;
  asm volatile("v_min_u32 %1, %2, %3\n\tv_min3_u32 %1, %1, %4, %5\n\tv_cmp_le_u32 %0, %6, %1"
               : "=s"(m), "=&v"(t) : "v"(g.f[0]), "v"(g.f[1]), "v"(g.f[2]), "v"(g.f[3]), "s"(g.call) : "memory");
  return m;
}
DI void wait_step(Gemm& g, int js) {
  unsigned long long lanes; unsigned t;
  asm volatile("s_cmp_eq_u32 %2, 0\n\ts_cbranch_scc1 .Ldone%=\n\t"
               "s_lshl_b64 %0, 3, %13\n"
               ".Lpoll%=:\n\t"
               "v_min_u32 %1, %6, %7\n\tv_min3_u32 %1, %1, %8, %9\n\tv_cmp_le_u32 vcc, %12, %1\n\t"
               "s_bitcmp1_b64 vcc, %13\n\ts_cbranch_scc1 .Ldone%=\n\t"
               "s_mov_b64 exec, %0\n\tbuffer_load_dwordx4 %5, %10, %11, 0 offen sc0 sc1\n\ts_mov_b64 exec, -1\n\t"
               "s_memrealtime vcc\n\ts_waitcnt lgkmcnt(0)\n\t"
               "s_sub_u32 vcc_lo, vcc_lo, %14\n\ts_cmp_lt_u32 vcc_lo, %15\n\ts_cbranch_scc1 .Lpoll%=\n\t"
               "s_mov_b32 %2, 0\n\ts_mov_b32 %3, 0\n\ts_mov_b32 %4, 0x7fc00000\n"
               ".Ldone%=:"
               : "=&s"(lanes), "=&v"(t), "+s"(g.need), "+s"(g.live), "+s"(g.scale)
               : "v"(g.f), "v"(g.f[0]), "v"(g.f[1]), "v"(g.f[2]), "v"(g.f[3]), "v"(g.vF), "s"(g.frs), "s"(g.call), "s"(js),
                 "s"(g.t0), "i"(DEADLINE) : "memory", "vcc", "scc");
}
#define AG_LD(k, o) "buffer_load_dwordx4 %" #k ", %17, %18, %16 offen offset:" #o " sc0 sc1\n\t"
#define AG_LD4(a, b, c, d) AG_LD(a, 0) AG_LD(b, 1024) AG_LD(c, 2048) AG_LD(d, 3072)
#define AG_NEXT "s_add_u32 %16, %16, 0x1000\n\t"
DI void load_block(Gemm& g, v4i (&A)[16], unsigned so, unsigned need) {
  asm volatile("s_cmp_eq_u32 %19, 0\n\ts_cbranch_scc1 .Lskip%=\n\t"
               AG_LD4(0, 1, 2, 3) AG_NEXT AG_LD4(4, 5, 6, 7) AG_NEXT AG_LD4(8, 9, 10, 11) AG_NEXT AG_LD4(12, 13, 14, 15) "\n.Lskip%=:"
               : "+v"(A[0]), "+v"(A[1]), "+v"(A[2]), "+v"(A[3]), "+v"(A[4]), "+v"(A[5]), "+v"(A[6]), "+v"(A[7]),
                 "+v"(A[8]), "+v"(A[9]), "+v"(A[10]), "+v"(A[11]), "+v"(A[12]), "+v"(A[13]), "+v"(A[14]), "+v"(A[15]), "+s"(so)
               : "v"(g.vR), "s"(g.ars), "s"(need) : "memory", "scc");
}
template <int O> DI void load_one(Gemm& g, v4i& a, unsigned so) {
  asm volatile("buffer_load_dwordx4 %0, %1, %2, %3 offen offset:%4 sc0 sc1" : "+v"(a) : "v"(g.vR), "s"(g.ars), "s"(so), "i"(O) : "memory");
}

template <int PB, bool END = false> DI void gemm_block(Gemm& g) {
  unsigned so, sst;
  unsigned long long m;
  f32x4 p[4], y, e;
  if constexpr (!END || !PB) wait_step(g, g.j2 - 1);
  GAP();
  if constexpr (!END || !PB) load_block(g, g.A[PB ^ 1], g.so1, g.need);
  GAP();
  static_for<0, 32>([&](auto tc) __attribute__((always_inline)) {
    constexpr int t = decltype(tc)::value, s = t >> 2, cb = t & 3;
    if constexpr (cb == 0) asm volatile("s_waitcnt vmcnt(%2)" : "+v"(g.A[PB][2 * s]), "+v"(g.A[PB][2 * s + 1]) : "i"(!END ? 33 : PB ? 15 - 2 * s : 33 - 2 * s) : "memory");
    const v8i a = __builtin_shufflevector(g.A[PB][2 * s], g.A[PB][2 * s + 1], 0, 1, 2, 3, 4, 5, 6, 7);
    g.acc[PB][cb] = __builtin_amdgcn_mfma_scale_f32_16x16x128_f8f6f4(g.W[s][cb], a, s == 0 ? f32x4{0.f, 0.f, 0.f, 0.f} : g.acc[PB][cb], 1, 0, 0, 0, 0, 0);
    GAP();
    if constexpr (!END && t == 0) m = ready(g);
    if constexpr (!END && t == 1) { g.lanes <<= 1; sample(g, g.lanes); }
    if constexpr (!END && t == 24) sample(g, g.lanes >> 1);
    if constexpr (!END && t == 2) {
      g.so1 = g.so2;
      asm volatile("s_bitcmp1_b64 %2, %3\n\ts_cselect_b32 %0, %4, %5\n\ts_cselect_b32 %1, 0, %6"
                   : "=s"(so), "=s"(g.need) : "s"(m), "s"(g.j2), "s"(g.so1), "i"(VOID_OFF), "s"(g.live) : "scc");
    }
    if constexpr (END && t == 28) g.j2 = pin_s(g.j2) + 1;
    if constexpr (!END && t == 28) { g.j2 = pin_s(g.j2) + 1; g.so2 = pin_s(__builtin_amdgcn_readlane(g.tab, g.j2)); }
    if constexpr (t == 2 || t == 4) {
      *(LAS f32x4*)(size_t)(g.wr[t - 2] + (PB ^ 1) * 4096) = g.acc[PB ^ 1][t - 2];
      *(LAS f32x4*)(size_t)(g.wr[t - 1] + (PB ^ 1) * 4096) = g.acc[PB ^ 1][t - 1];
    }
    if constexpr (!END && t >= 3 && (t & 1)) {
      constexpr int i = (t - 3) >> 1;
      if constexpr (i && !(i & 3)) so = pin_s(so) + 4096;
      load_one<(i & 3) * 1024>(g, g.A[PB][i], so);
      if constexpr (t == 31) load_one<3072>(g, g.A[PB][15], so);
    }
    if constexpr (t == 8) { epi_barrier(); p[0] = *(LAS f32x4*)(size_t)(g.rd0 + (PB ^ 1) * 4096); p[1] = *(LAS f32x4*)(size_t)(g.rd0 + (PB ^ 1) * 4096 + 32768); }
    if constexpr (t == 10) { p[2] = *(LAS f32x4*)(size_t)(g.rd1 + (PB ^ 1) * 4096); p[3] = *(LAS f32x4*)(size_t)(g.rd1 + (PB ^ 1) * 4096 + 32768); }
    if constexpr (t == 12) { _Pragma("unroll") for (int i = 0; i < 4; ++i) p[0][i] = fadd(p[0][i], p[1][i]); }
    if constexpr (t == 14) { _Pragma("unroll") for (int i = 0; i < 4; ++i) p[2][i] = fadd(p[2][i], p[3][i]); }
    if constexpr (t == 15) { _Pragma("unroll") for (int i = 0; i < 4; ++i) p[0][i] = fadd(p[0][i], p[2][i]); }
    if constexpr (t == 16) { _Pragma("unroll") for (int i = 0; i < 4; ++i) y[i] = fmuls(p[0][i], g.scale); }
    if constexpr (t >= 17 && t <= 20) e[t - 17] = silu_a(y[t - 17]);
    if constexpr (t >= 22 && t <= 25) y[t - 22] = fmul(y[t - 22], e[t - 22]);
    if constexpr (t >= 21 && t <= 24) e[t - 21] = silu_b(e[t - 21]);
    if constexpr (t == 6) sst = pin_s(__builtin_amdgcn_readlane(g.tab, pin_s(g.j2) - 3) * 2 + g.col4);
    if constexpr (t == 26) epi_store(g, sst, y);
    GAP();
  });
}
template <int PB> DI void gemm_tail(Gemm& g) {
  f32x4 p[4], y;
  epi_write(g, PB); epi_barrier(); epi_read(g, PB, p);
#pragma unroll
  for (int i = 0; i < 4; ++i) {
    y[i] = ((p[0][i] + p[1][i]) + (p[2][i] + p[3][i])) * g.scale;
    y[i] *= silu_b(silu_a(y[i]));
  }
  epi_store(g, gblk(g, g.nblk - 1) * (2 * BLK) + g.col4, y);
}

#define AG_PARK "s_waitcnt vmcnt(%5)\n\tds_write_b128 %0, %1\n\tds_write_b128 %0, %2 offset:1024\n\tds_write_b128 %0, %3 offset:2048\n\tds_write_b128 %0, %4 offset:3072"
template <int N, bool ACC> DI void w_park(unsigned a, const v4i* w) {
  if constexpr (ACC) asm volatile(AG_PARK :: "v"(a), "a"(w[0]), "a"(w[1]), "a"(w[2]), "a"(w[3]), "i"(N) : "memory");
  else asm volatile(AG_PARK :: "v"(a), "v"(w[0]), "v"(w[1]), "v"(w[2]), "v"(w[3]), "i"(N) : "memory");
}

__global__ void __launch_bounds__(256, 1) ag_gemm(Params p) {
  extern __shared__ __attribute__((aligned(16))) u8 smem[];
  const unsigned lane = threadIdx.x & 63;
  const unsigned wave = __builtin_amdgcn_readfirstlane(threadIdx.x >> 6);
  const unsigned b = blockIdx.x;
  const unsigned link = (b & 4) | ((b + (b >> 3)) & 3);
  u8* const dws = p.peer[link];
  const unsigned mapw = p.map[b >> 2];
  const unsigned pc = pin_s((b & 3) * NPW), pg = pin_s(((b - NCOMM) & 127) * 4), piece = (b < NCOMM ? pc : pg) + wave, prb = piece >> 4, psub = piece & 15;
  unsigned xo = prb * BLK + psub * 256 + (lane & 15) * KD + (lane >> 4) * 16, co = CNT_OFF + b * 128 + wave * 4;
  asm volatile("" : "+v"(xo), "+v"(co));
  asm volatile("" :: "s"(p.x), "s"(p.w), "s"(p.sx), "s"(p.sw), "s"(p.out), "s"(p.myws), "s"(p.rank), "s"(p.ranks), "s"(dws), "s"(mapw));
  u32x4 xs[4];
  unsigned callv;
  asm volatile("global_load_dwordx4 %0, %5, %7\n\tglobal_load_dwordx4 %1, %5, %7 offset:64\n\tglobal_load_dwordx4 %2, %5, %7 offset:128\n\t"
               "global_load_dwordx4 %3, %5, %7 offset:192\n\tglobal_load_dword %4, %6, %8 sc1"
               : "=&v"(xs[0]), "=&v"(xs[1]), "=&v"(xs[2]), "=&v"(xs[3]), "=&v"(callv) : "v"(xo), "v"(co), "s"(p.x), "s"(p.myws) : "memory");
  unsigned* cnt = (unsigned*)(p.myws + CNT_OFF + b * 128 + wave * 4);

  if (b < NCOMM && dws != p.myws) {
    const __amdgpu_buffer_rsrc_t xrs = __builtin_amdgcn_make_buffer_rsrc((void*)p.x, 0, SHARD, 0x00020000);
    const __amdgpu_buffer_rsrc_t frs = __builtin_amdgcn_make_buffer_rsrc((void*)(dws + FLAG_OFF + p.rank * FLAG_SRC), 0, FLAG_SRC, 0x00020000);
    const __amdgpu_buffer_rsrc_t mrs = __builtin_amdgcn_make_buffer_rsrc((void*)p.myws, 0, TICK_OFF, 0x00020000);
    const unsigned vx = (lane & 15) * KD + (lane >> 4) * 16, vd = lane * 16;
    const unsigned peer = (p.ranks >> 4 * link) & 7;
    u32x4 d[4];
    asm volatile("buffer_load_dwordx4 %0, %4, %5, %6 offen\n\tbuffer_load_dwordx4 %1, %4, %5, %6 offen offset:64\n\t"
                 "buffer_load_dwordx4 %2, %4, %5, %6 offen offset:128\n\tbuffer_load_dwordx4 %3, %4, %5, %6 offen offset:192"
                 : "=&v"(d[0]), "=&v"(d[1]), "=&v"(d[2]), "=&v"(d[3])
                 : "v"(vx), "s"(xrs), "s"(((piece + 4 * NPW) >> 4) * BLK + ((piece + 4 * NPW) & 15) * 256) : "memory");
    unsigned warm;
    asm volatile("buffer_load_dword %0, %1, %2, 0 offen sc0 sc1" : "=v"(warm)
                 : "v"((unsigned)FLAG_OFF + peer * FLAG_SRC + (((b & 3) * 4 + wave) * 8 + (lane & 7)) * FQ), "s"(mrs) : "memory");
    const unsigned ring = (unsigned)(size_t)smem;
    if (wave == NPW) *(LAS unsigned*)(size_t)(ring + lane * 4) = VOID_OFF;
    asm volatile("s_waitcnt lgkmcnt(0)\n\ts_barrier" ::: "memory");
    unsigned call;
    asm volatile("s_waitcnt vmcnt(5)\n\tv_readfirstlane_b32 %0, %1" : "=s"(call) : "v"(callv) : "memory");
    const unsigned base = call * 512 - 16 * NPW;
    call += 1;
    unsigned callx = call;
    asm volatile("" : "+v"(callx));
    if (wave == NPW) {
      asm volatile("s_waitcnt vmcnt(0)" :: "v"(d[0]), "v"(d[1]), "v"(d[2]), "v"(d[3]) : "memory");
      unsigned seen = VOID_OFF, v, last = 0;
      asm volatile(".Lscan%=:\n\tds_read_b32 %[v], %[a]\n\ts_waitcnt lgkmcnt(0)\n\t"
                   "v_cmp_ne_u32 vcc, %[v], %[seen]\n\ts_cbranch_vccz .Lsame%=\n\t"
                   "s_mov_b64 exec, vcc\n\tbuffer_store_dword %[cx], %[v], %[frs], 0 offen sc0 sc1\n\ts_mov_b64 exec, -1\n\tv_mov_b32 %[seen], %[v]\n"
                   ".Lsame%=:\n\ts_cmp_eq_u32 %[last], 7\n\ts_cbranch_scc1 .Lout%=\n\t"
                   "s_lshr_b32 vcc_lo, vcc_hi, 16\n\ts_or_b32 %[last], %[last], vcc_lo\n\ts_branch .Lscan%=\n"
                   ".Lout%=:\n\ts_waitcnt vmcnt(0)"
                   : [seen] "+&v"(seen), [v] "=&v"(v), [last] "+&s"(last)
                   : [a] "v"(ring + lane * 4), [cx] "v"(callx), [frs] "s"(frs) : "memory", "vcc", "scc");
    } else {
      const __amdgpu_buffer_rsrc_t drs = __builtin_amdgcn_make_buffer_rsrc((void*)(dws + DATA_OFF + (size_t)((call & 1) * NR + p.rank) * SHARD), 0, SHARD, 0x00020000);
      const __amdgpu_buffer_rsrc_t trs = __builtin_amdgcn_make_buffer_rsrc((void*)(p.myws + TICK_OFF + link * 128), 0, 128, 0x00020000);
      unsigned tka = base + 8 * NPW + piece, tkb = base + 12 * NPW + piece;
      unsigned ta = piece, tb = 4 * NPW + piece, fo = VOID_OFF, ra = ring + wave * 64, s1, s2, v1, v2;
      asm volatile("" : "+v"(tka), "+v"(tkb));
#define AG_PS(d, o) "buffer_store_dwordx4 %[" d "], %[vd], %[drs], %[s1] offen offset:" #o " sc0 sc1\n\t"
#define AG_XL(d, o) "buffer_load_dwordx4 %[" d "], %[vx], %[xrs], %[s1] offen offset:" #o "\n\t"
#define AG_STEP(t, tk, d0, d1, d2, d3) \
      "s_lshl_b32 %[s1], %[" t "], 12\n\t" \
      "s_waitcnt vmcnt(8)\n\t" AG_PS(d0, 0) "s_waitcnt vmcnt(8)\n\t" AG_PS(d1, 1024) "s_waitcnt vmcnt(8)\n\t" AG_PS(d2, 2048) \
      "s_waitcnt vmcnt(8)\n\t" AG_PS(d3, 3072) \
      "s_mov_b64 exec, 1\n\tv_mov_b32 %[v1], %[fo]\n\tv_mov_b32 %[v2], %[ra]\n\tds_write_b32 %[v2], %[v1]\n\ts_mov_b64 exec, -1\n\t" \
      "s_add_u32 %[ra], %[ra], 4\n\ts_and_b32 %[ra], %[ra], 63\n\ts_or_b32 %[ra], %[ra], %[rb]\n\t" \
      "s_lshr_b32 %[fo], %[" t "], 2\n\ts_lshl_b32 %[fo], %[fo], 7\n\ts_and_b32 %[s2], %[" t "], 3\n\ts_lshl2_add_u32 %[fo], %[s2], %[fo]\n\t" \
      "v_readfirstlane_b32 %[" t "], %[" tk "]\n\ts_sub_u32 %[" t "], %[" t "], %[base]\n\t" \
      "s_lshr_b32 %[s1], %[" t "], 4\n\ts_lshl_b32 %[s1], %[s1], 8\n\ts_and_b32 %[s2], %[" t "], 15\n\ts_or_b32 %[s1], %[s1], %[s2]\n\ts_lshl_b32 %[s1], %[s1], 8\n\t" \
      AG_XL(d0, 0) AG_XL(d1, 64) AG_XL(d2, 128) AG_XL(d3, 192) \
      "s_mov_b64 exec, 1\n\tv_mov_b32 %[" tk "], 1\n\tbuffer_atomic_add %[" tk "], off, %[trs], 0 sc0 sc1\n\ts_mov_b64 exec, -1\n\t"
      asm volatile(".Lpush%=:\n\t" AG_STEP("ta", "tka", "a0", "a1", "a2", "a3") "s_cmpk_gt_u32 %[tb], 511\n\ts_cbranch_scc1 .Lend%=\n\t"
                   AG_STEP("tb", "tkb", "b0", "b1", "b2", "b3") "s_cmpk_lt_u32 %[ta], 512\n\ts_cbranch_scc1 .Lpush%=\n"
                   ".Lend%=:\n\tv_mov_b32 %[v1], %[vo]+4\n\tv_mov_b32 %[v2], %[rd]\n\tds_write_b32 %[v2], %[v1]\n\t"
                   "s_waitcnt vmcnt(5)\n\ts_mov_b64 exec, 1\n\tbuffer_store_dword %[cx], off, %[frs], %[fo] sc0 sc1\n\ts_mov_b64 exec, -1\n\t"
                   "s_waitcnt vmcnt(0) lgkmcnt(0)"
                   : [ta] "+&s"(ta), [tb] "+&s"(tb), [fo] "+&s"(fo), [ra] "+&s"(ra), [s1] "=&s"(s1), [s2] "=&s"(s2), [v1] "=&v"(v1), [v2] "=&v"(v2)
                   : [a0] "v"(xs[0]), [a1] "v"(xs[1]), [a2] "v"(xs[2]), [a3] "v"(xs[3]), [b0] "v"(d[0]), [b1] "v"(d[1]), [b2] "v"(d[2]), [b3] "v"(d[3]),
                     [tka] "v"(tka), [tkb] "v"(tkb), [cx] "v"(callx), [vx] "v"(vx), [vd] "v"(vd),
                     [xrs] "s"(xrs), [frs] "s"(frs), [trs] "s"(trs), [drs] "s"(drs), [base] "s"(base),
                     [rb] "s"(ring + wave * 64), [rd] "s"(ring + 192 + wave * 4), [vo] "i"(VOID_OFF)
                   : "memory", "scc");
    }
    asm volatile("" :: "v"(warm));
    if (lane == 0) asm volatile("global_store_dword %0, %1, off sc1" :: "v"(cnt), "v"(call) : "memory");
    return;
  }

  const unsigned gi = b - NCOMM, me = mapw >> (b & 3) * 8, rs = (me >> 5) & 7, col0 = (me & 31) * 64;
  const unsigned srcp = ((unsigned)p.rank + 1 + rs) % NR;
  const unsigned lds = (unsigned)(size_t)smem + wave * 32768;
  Gemm g;
  g.t0 = (unsigned)__builtin_amdgcn_s_memrealtime();
  __amdgpu_buffer_rsrc_t wrs = __builtin_amdgcn_make_buffer_rsrc((void*)p.w, 0, KD * NL, 0x00020000);
  const unsigned wv0 = (lane >> 2) * NL + (((lane & 3) ^ ((lane >> 4) & 1)) * 16), wv1 = wv0 ^ 32;
  const unsigned ws0 = wave * 1024 * NL + col0;
  const unsigned rot = rs * 4;
  auto w_dma = [&](int j0) {
#pragma unroll
    for (int j = j0; j < j0 + 16; ++j) {
      const unsigned q = (j + rot) & 31;
      asm volatile("s_mov_b32 m0, %0\n\ts_nop 0\n\tbuffer_load_dwordx4 %1, %2, %3 offen lds"
                   :: "s"(lds + q * 1024), "v"((j & 1) ? wv1 : wv0), "s"(wrs), "s"(ws0 + q * 16 * NL) : "memory");
    }
  };
  const unsigned tg = lane >> 4, tr = (lane & 15) >> 1, tf = ((tr >> 2) & 1) | ((tg & 1) << 1);
  const unsigned tb = lds + (tg * 16 + tr) * 64 + tf * 16 + (lane & 1) * 8;
  auto w_read = [&](int r) {
#pragma unroll
    for (int h = 0; h < 2; ++h)
#pragma unroll
      for (int cb = 0; cb < 4; ++cb) {
        v2i t[4];
#pragma unroll
        for (int k = 0; k < 4; ++k)
          t[k] = __builtin_amdgcn_ds_read_tr8_b64_v2i32((LAS v2i*)(size_t)((tb ^ (cb * 16)) + (r & 1) * 16384 + (h * 128 + (k >> 1) * 64 + (k & 1) * 8) * 64));
        g.W[2 * r + h][cb] = v8i{t[0][0], t[0][1], t[1][0], t[1][1], t[2][0], t[2][1], t[3][0], t[3][1]};
      }
#pragma unroll
    for (int h = 0; h < 2; ++h)
#pragma unroll
      for (int cb = 0; cb < 4; ++cb) {
        v8i& f = g.W[2 * r + h][cb];
        if ((2 * r + h) * 4 + cb < 27) asm volatile("" : "+a"(f)); else asm volatile("" : "+v"(f));
      }
  };
  w_dma(0);
  g.ors = __builtin_amdgcn_make_buffer_rsrc((void*)p.out, 0, OUT_BYTES, 0x00020000);
  g.frs = __builtin_amdgcn_make_buffer_rsrc((void*)p.myws, 0, TICK_OFF, 0x00020000);
  g.vR = lane * 16 + wave * 16384;
  g.vO = (wave * 4 + (lane >> 4)) * (NL * 4) + (lane & 15) * 16;
#pragma unroll
  for (int cb = 0; cb < 4; ++cb) g.wr[cb] = lds + (lane & 15) * 256 + (((cb * 4 + (lane >> 4)) ^ (lane & 15)) * 16);
  {
    const unsigned mr = wave * 4 + (lane >> 4);
    g.rd0 = (unsigned)(size_t)smem + mr * 256 + (((lane & 15) ^ mr) * 16);
    g.rd1 = g.rd0 + 65536;
    asm volatile("" : "+v"(g.rd1));
  }
  g.nl = rs == 7 ? NRB : (((unsigned)(size_t)p.w + col0) >> 7 & 7) == 3 ? 0 : (int)(LOCAL0 >> 8 * (rs + 1) & 63) - (int)(LOCAL0 >> 8 * rs & 63);
  g.nblk = rs == 7 ? NRB : g.nl + NRB;
  g.blk_l = p.rank * NRB + (rs < 7 ? (int)(LOCAL0 >> 8 * rs & 63) : 0);
  g.blk_r = srcp * NRB - g.nl;
  g.col4 = col0 * 4;
  g.vF = FLAG_OFF + gblk(g, (int)lane < g.nblk ? lane : 0) * FB + wave * FQ;
  g.tab = (int)lane < g.nblk ? gblk(g, lane) * BLK : VOID_OFF;
  g.f = u32x4{0, 0, 0, 0};
  asm volatile("" : "+v"(g.f));
  g.scale = __builtin_bit_cast(float, __builtin_amdgcn_readfirstlane(__builtin_bit_cast(int,
      *(const __attribute__((address_space(4))) float*)p.sx * *(const __attribute__((address_space(4))) float*)p.sw)));
  g.live = 1; g.need = 1;
  waitv<16>();
  asm volatile("v_readfirstlane_b32 %0, %1" : "=s"(g.call) : "v"(callv) : "memory");
  g.call += 1;
  g.slots = p.myws + DATA_OFF + (size_t)((g.call & 1) * NR) * SHARD;
  g.ars = __builtin_amdgcn_make_buffer_rsrc((void*)g.slots, 0, NR * SHARD, 0x00020000);
  const unsigned call0 = g.call;
  {
    __amdgpu_buffer_rsrc_t prs = __builtin_amdgcn_make_buffer_rsrc((void*)g.slots, 0, gi < 128 ? NR * SHARD : 0u, 0x00020000);
#pragma unroll
    for (int c = 0; c < 4; ++c)
      asm volatile("buffer_store_dwordx4 %0, %1, %2, %3 offen sc0 sc1\n\ts_nop 1"
                   :: "v"(xs[c]), "v"(lane * 16 + c * 1024), "s"(prs), "s"((p.rank * NRB + prb) * BLK + psub * 4096) : "memory");
  }
  w_dma(16);
  waitv<16>();
  {
    __amdgpu_buffer_rsrc_t prs = __builtin_amdgcn_make_buffer_rsrc((void*)p.myws, 0, gi < 128 ? TICK_OFF : 0u, 0x00020000);
    unsigned callx = g.call;
    asm volatile("" : "+v"(callx));
    asm volatile("s_mov_b64 exec, 1\n\tbuffer_store_dword %0, off, %1, %2 sc0 sc1\n\ts_mov_b64 exec, -1"
                 :: "v"(callx), "s"(prs), "s"((unsigned)FLAG_OFF + (p.rank * NRB + prb) * FB + (psub >> 2) * FQ + (psub & 3) * 4) : "memory");
  }
  v4i wr[32];
#pragma unroll
  for (int j = 0; j < 32; ++j) {
    const unsigned so = ws0 + (32 + ((j + pin_s(rot)) & 31)) * 16 * NL;
    if (j < 24) asm volatile("buffer_load_dwordx4 %0, %1, %2, %3 offen" : "=a"(wr[j]) : "v"((j & 1) ? wv1 : wv0), "s"(wrs), "s"(so) : "memory");
    else asm volatile("buffer_load_dwordx4 %0, %1, %2, %3 offen" : "=v"(wr[j]) : "v"((j & 1) ? wv1 : wv0), "s"(wrs), "s"(so) : "memory");
    if (j == 15 || j == 27) sample(g, 15);
  }
  waitv<34>();
  w_read(0); w_read(1);
  wait_step(g, 0);
  wait_step(g, 1);
  g.lanes = 3ull << 2;
  sample(g, g.lanes);
#pragma unroll
  for (int i = 0; i < 16; ++i) { asm volatile("" : "=v"(g.A[0][i])); asm volatile("" : "=v"(g.A[1][i])); }
  int odd = g.nblk & 1;
  load_block(g, g.A[1], g.live ? gblk(g, 0) * BLK : VOID_OFF, odd);
  load_block(g, g.A[0], g.live ? gblk(g, odd) * BLK : VOID_OFF, 1);
  load_block(g, g.A[1], g.live ? gblk(g, 1) * BLK : VOID_OFF, odd ^ 1);
  static_for<0, 8>([&](auto c) __attribute__((always_inline)) {
    constexpr int j = decltype(c)::value * 4;
    w_park<61 - j, (j < 24)>(lds + ((j + pin_s(rot)) & 31) * 1024 + lane * 16, wr + j);
  });
#pragma unroll
  for (int cb = 0; cb < 4; ++cb) asm volatile("" : "=a"(g.acc[0][cb]), "=a"(g.acc[1][cb]));
  g.need = 0; g.so1 = 0; g.so2 = gblk(g, 2) * BLK; g.j2 = 2;
  w_read(2); w_read(3);
  waitv<16>();
  asm volatile("s_mov_b64 exec, 1\n\tglobal_store_dword %0, %1, off sc1\n\ts_mov_b64 exec, -1" :: "v"(cnt), "v"(call0) : "memory");
  GAP();
  const int jend = g.nblk;
  do {
    if (!odd) gemm_block<0>(g);
    odd = pin_s(0);
    gemm_block<1>(g);
  } while (g.j2 < jend);
  gemm_block<0, true>(g);
  gemm_block<1, true>(g);
  gemm_tail<1>(g);
  if (!g.live) asm volatile("global_store_dword %0, %1, off sc1" :: "v"(p.myws + TMO_OFF), "v"(g.call) : "memory");
  asm volatile("" :: "v"(g.f));
#pragma unroll
  for (int i = 0; i < 16; ++i) asm volatile("" :: "v"(g.A[0][i]), "v"(g.A[1][i]));
}

}

#define HIP_CHECK(expr) do { hipError_t _e = (expr); if (_e != hipSuccess) { \
    fprintf(stderr, "HIP error %s at %s:%d\n", hipGetErrorString(_e), __FILE__, __LINE__); \
    return; } } while (0)


extern "C" void kernel_launch(int rank, int nranks, ncclComm_t comm, hipStream_t stream,
                              void* const* d_in, const int* in_sizes, int n_in,
                              void* d_out, int out_size, const int* in_dtypes, int out_dtype,
                              void* const* d_out_all, void* const* d_ws_all, size_t ws_bytes) {
  (void)comm; (void)in_dtypes; (void)out_dtype; (void)d_out_all;
  if (nranks != NR || (unsigned)rank >= (unsigned)NR || n_in < 4 || ws_bytes < WS_NEED ||
      in_sizes[0] != (int)(MSH * KD) || in_sizes[1] != (int)(KD * NL) || out_size != (int)(4096u * NL)) {
    fprintf(stderr, "kernel_launch: unexpected problem shape\n");
    return;
  }
  HIP_CHECK(hipFuncSetAttribute((const void*)ag_gemm, hipFuncAttributeMaxDynamicSharedMemorySize, LDS_BYTES));
  Params p;
  p.x = (const u8*)d_in[0]; p.w = (const u8*)d_in[1];
  p.sx = (const float*)d_in[2]; p.sw = (const float*)d_in[3];
  p.out = (float*)d_out;
  p.ranks = 0;
  for (int i = 0, lo = 0, hi = 4; i < NR; ++i) {
    const int r = (rank + 1 + i) % NR;
    const int x = (((GPU_SET >> r) ^ (GPU_SET >> rank)) & 1) ? hi++ : lo++;
    p.peer[x] = (u8*)d_ws_all[r];
    p.ranks |= r << 4 * x;
  }
  u8 map[GRID] = {};
  int cgs[32], ns = 0, nf = 0;
  for (int cg = 0; cg < 32; ++cg) {
    if ((((size_t)p.w + cg * 64) >> 7 & 7) == 3) { cgs[(ns >> 1) * 8 + 6 + (ns & 1)] = cg; map[24 - 7 * ns] = cg | 7 << 5; ++ns; }
    else { cgs[nf < 12 ? nf / 6 * 8 + nf % 6 : nf + 4] = cg; ++nf; }
  }
  for (int x = 0; x < 8; ++x)
    for (int late = 0, n = 0; late < 2; ++late)
      for (int i = 28 * (x & 1); i < 28 * (x & 1) + 28; ++i)
        if ((x < 4 && (i & 7) >= 6) == late) map[NCOMM + n++ * 8 + x] = cgs[(x >> 1) * 8 + (i & 7)] | (i >> 3) << 5;
  for (int i = 0; i < GRID / 4; ++i) p.map[i] = map[4 * i] | map[4 * i + 1] << 8 | map[4 * i + 2] << 16 | (unsigned)map[4 * i + 3] << 24;
  p.myws = (u8*)d_ws_all[rank];
  p.rank = rank;
  hipLaunchKernelGGL(ag_gemm, dim3(GRID), dim3(256), LDS_BYTES, stream, p);
  HIP_CHECK(hipGetLastError());
}
```
